```python
import numpy as np

kernel_name = "gpu_a2a_gemm_m4096_k4096_n16384_e5m2e5m2_dqrelu"


ABSMAX_RTOL = 0.05

COMPUTE_DTYPE = "float32"
EPILOGUE = "dequant_relu"
IN_QUANT = None
OUT_QUANT = None
IN_FP8 = "e5m2e5m2"
IN_DTYPES = {"x": "e5m2", "w_mat": "e5m2",
             "scale_x": "float32", "scale_w": "float32"}
WORLD_SIZE = 8

def _snap_fp8(x, mant_bits, max_abs, min_normal):
    a = np.abs(x.astype(np.float64))
    _m, _e = np.frexp(a)
    q = np.where(a >= min_normal, np.ldexp(1.0, _e - 1 - mant_bits),
                 min_normal * 2.0 ** -mant_bits)
    snapped = np.minimum(np.round(a / q) * q, max_abs)
    return (np.sign(x) * snapped).astype(np.float32)

def setup_inputs():

    x = _snap_fp8(np.random.randn(4096, 4096).astype(np.float32), 2, 57344.0, 2.0 ** -14)
    w_mat = _snap_fp8(np.random.randn(4096, 16384).astype(np.float32), 2, 57344.0, 2.0 ** -14)
    scale_x = (0.02 * (1.0 + np.random.rand(1))).astype(np.float32)
    scale_w = (0.02 * (1.0 + np.random.rand(1))).astype(np.float32)
    return {"x": x, "w_mat": w_mat, "scale_x": scale_x, "scale_w": scale_w}

def reference(x, w_mat, scale_x, scale_w):
    acc = (x @ w_mat).astype(np.float32)
    y = (acc * (scale_x * scale_w)).astype(np.float32)
    return np.maximum(y, 0.0).astype(np.float32)
```

```cpp
#define COMPUTE_ASM \
    "v_mov_b32 v24, %[xa]\n\t" \
    "v_xor_b32 v25, 64, %[xa]\n\t" \
    "v_mov_b32 v26, %[wb]\n\t" \
    "v_xor_b32 v27, 16, %[wb]\n\t" \
    "v_xor_b32 v28, 32, %[wb]\n\t" \
    "v_xor_b32 v29, 48, %[wb]\n\t" \
    "v_mov_b32 v16, %[voff]\n\t" \
    "v_add_u32 v17, 0x100000, v16\n\t" \
    "v_add_u32 v18, 0x100000, v17\n\t" \
    "v_add_u32 v19, 0x100000, v18\n\t" \
    "v_add_u32 v20, 0x100000, v19\n\t" \
    "v_add_u32 v21, 0x100000, v20\n\t" \
    "v_add_u32 v22, 0x100000, v21\n\t" \
    "v_add_u32 v23, 0x100000, v22\n\t" \
    "v_mov_b64 v[128:129], 0\n\t" \
    "v_mov_b64 v[130:131], 0\n\t" \
    "v_mov_b64 v[132:133], 0\n\t" \
    "v_mov_b64 v[134:135], 0\n\t" \
    "v_mov_b64 v[136:137], 0\n\t" \
    "v_mov_b64 v[138:139], 0\n\t" \
    "v_mov_b64 v[140:141], 0\n\t" \
    "v_mov_b64 v[142:143], 0\n\t" \
    "v_mov_b64 v[144:145], 0\n\t" \
    "v_mov_b64 v[146:147], 0\n\t" \
    "v_mov_b64 v[148:149], 0\n\t" \
    "v_mov_b64 v[150:151], 0\n\t" \
    "v_mov_b64 v[152:153], 0\n\t" \
    "v_mov_b64 v[154:155], 0\n\t" \
    "v_mov_b64 v[156:157], 0\n\t" \
    "v_mov_b64 v[158:159], 0\n\t" \
    "v_mov_b64 v[160:161], 0\n\t" \
    "v_mov_b64 v[162:163], 0\n\t" \
    "v_mov_b64 v[164:165], 0\n\t" \
    "v_mov_b64 v[166:167], 0\n\t" \
    "v_mov_b64 v[168:169], 0\n\t" \
    "v_mov_b64 v[170:171], 0\n\t" \
    "v_mov_b64 v[172:173], 0\n\t" \
    "v_mov_b64 v[174:175], 0\n\t" \
    "v_mov_b64 v[176:177], 0\n\t" \
    "v_mov_b64 v[178:179], 0\n\t" \
    "v_mov_b64 v[180:181], 0\n\t" \
    "v_mov_b64 v[182:183], 0\n\t" \
    "v_mov_b64 v[184:185], 0\n\t" \
    "v_mov_b64 v[186:187], 0\n\t" \
    "v_mov_b64 v[188:189], 0\n\t" \
    "v_mov_b64 v[190:191], 0\n\t" \
    "v_mov_b64 v[192:193], 0\n\t" \
    "v_mov_b64 v[194:195], 0\n\t" \
    "v_mov_b64 v[196:197], 0\n\t" \
    "v_mov_b64 v[198:199], 0\n\t" \
    "v_mov_b64 v[200:201], 0\n\t" \
    "v_mov_b64 v[202:203], 0\n\t" \
    "v_mov_b64 v[204:205], 0\n\t" \
    "v_mov_b64 v[206:207], 0\n\t" \
    "v_mov_b64 v[208:209], 0\n\t" \
    "v_mov_b64 v[210:211], 0\n\t" \
    "v_mov_b64 v[212:213], 0\n\t" \
    "v_mov_b64 v[214:215], 0\n\t" \
    "v_mov_b64 v[216:217], 0\n\t" \
    "v_mov_b64 v[218:219], 0\n\t" \
    "v_mov_b64 v[220:221], 0\n\t" \
    "v_mov_b64 v[222:223], 0\n\t" \
    "v_mov_b64 v[224:225], 0\n\t" \
    "v_mov_b64 v[226:227], 0\n\t" \
    "v_mov_b64 v[228:229], 0\n\t" \
    "v_mov_b64 v[230:231], 0\n\t" \
    "v_mov_b64 v[232:233], 0\n\t" \
    "v_mov_b64 v[234:235], 0\n\t" \
    "v_mov_b64 v[236:237], 0\n\t" \
    "v_mov_b64 v[238:239], 0\n\t" \
    "v_mov_b64 v[240:241], 0\n\t" \
    "v_mov_b64 v[242:243], 0\n\t" \
    "v_mov_b64 v[244:245], 0\n\t" \
    "v_mov_b64 v[246:247], 0\n\t" \
    "v_mov_b64 v[248:249], 0\n\t" \
    "v_mov_b64 v[250:251], 0\n\t" \
    "v_mov_b64 v[252:253], 0\n\t" \
    "v_mov_b64 v[254:255], 0\n\t" \
    "s_mov_b32 %[st], 0\n\t" \
    "s_mov_b32 %[cnt], 31\n\t" \
    "s_mov_b32 %[cpos], 49152\n\t" \
    "s_mov_b32 %[cneg], -98304\n\t" \
    "s_barrier\n\t" \
    "ds_read_b64_tr_b8 v[96:97], v26 offset:0\n\t" \
    "ds_read_b64_tr_b8 v[98:99], v26 offset:1024\n\t" \
    "ds_read_b64_tr_b8 v[100:101], v26 offset:8192\n\t" \
    "ds_read_b64_tr_b8 v[102:103], v26 offset:9216\n\t" \
    "ds_read_b128 v[32:35], v24 offset:0\n\t" \
    "ds_read_b128 v[36:39], v25 offset:0\n\t" \
    "ds_read_b64_tr_b8 v[104:105], v27 offset:0\n\t" \
    "ds_read_b64_tr_b8 v[106:107], v27 offset:1024\n\t" \
    "ds_read_b64_tr_b8 v[108:109], v27 offset:8192\n\t" \
    "ds_read_b64_tr_b8 v[110:111], v27 offset:9216\n\t" \
    "ds_read_b128 v[40:43], v24 offset:2048\n\t" \
    "ds_read_b128 v[44:47], v25 offset:2048\n\t" \
    "ds_read_b128 v[48:51], v24 offset:4096\n\t" \
    "ds_read_b128 v[52:55], v25 offset:4096\n\t" \
    "s_add_u32 %[st], %[st], 1\n\t" \
    "s_cmp_eq_u32 %[st], 3\n\t" \
    "s_cselect_b32 %[dlt], %[cneg], %[cpos]\n\t" \
    "s_cselect_b32 %[st], 0, %[st]\n\t" \
    "s_waitcnt lgkmcnt(8)\n\t" \
    "v_mfma_f32_16x16x128_f8f6f4 v[128:131], v[96:103], v[32:39], v[128:131] cbsz:1 blgp:1\n\t" \
    "ds_read_b128 v[56:59], v24 offset:6144\n\t" \
    "ds_read_b128 v[60:63], v25 offset:6144\n\t" \
    "s_waitcnt lgkmcnt(6)\n\t" \
    "v_mfma_f32_16x16x128_f8f6f4 v[132:135], v[104:111], v[32:39], v[132:135] cbsz:1 blgp:1\n\t" \
    "ds_read_b128 v[64:67], v24 offset:8192\n\t" \
    "ds_read_b128 v[68:71], v25 offset:8192\n\t" \
    "s_waitcnt lgkmcnt(6)\n\t" \
    "v_mfma_f32_16x16x128_f8f6f4 v[148:151], v[104:111], v[40:47], v[148:151] cbsz:1 blgp:1\n\t" \
    "ds_read_b128 v[72:75], v24 offset:10240\n\t" \
    "ds_read_b128 v[76:79], v25 offset:10240\n\t" \
    "v_mfma_f32_16x16x128_f8f6f4 v[144:147], v[96:103], v[40:47], v[144:147] cbsz:1 blgp:1\n\t" \
    "ds_read_b128 v[80:83], v24 offset:12288\n\t" \
    "ds_read_b128 v[84:87], v25 offset:12288\n\t" \
    "s_waitcnt lgkmcnt(8)\n\t" \
    "v_mfma_f32_16x16x128_f8f6f4 v[160:163], v[96:103], v[48:55], v[160:163] cbsz:1 blgp:1\n\t" \
    "ds_read_b128 v[88:91], v24 offset:14336\n\t" \
    "ds_read_b128 v[92:95], v25 offset:14336\n\t" \
    "v_mfma_f32_16x16x128_f8f6f4 v[164:167], v[104:111], v[48:55], v[164:167] cbsz:1 blgp:1\n\t" \
    "ds_read_b64_tr_b8 v[112:113], v28 offset:0\n\t" \
    "ds_read_b64_tr_b8 v[114:115], v28 offset:1024\n\t" \
    "s_waitcnt lgkmcnt(10)\n\t" \
    "v_mfma_f32_16x16x128_f8f6f4 v[180:183], v[104:111], v[56:63], v[180:183] cbsz:1 blgp:1\n\t" \
    "ds_read_b64_tr_b8 v[116:117], v28 offset:8192\n\t" \
    "ds_read_b64_tr_b8 v[118:119], v28 offset:9216\n\t" \
    "v_mfma_f32_16x16x128_f8f6f4 v[176:179], v[96:103], v[56:63], v[176:179] cbsz:1 blgp:1\n\t" \
    "ds_read_b64_tr_b8 v[120:121], v29 offset:0\n\t" \
    "ds_read_b64_tr_b8 v[122:123], v29 offset:1024\n\t" \
    "s_waitcnt lgkmcnt(12)\n\t" \
    "v_mfma_f32_16x16x128_f8f6f4 v[192:195], v[96:103], v[64:71], v[192:195] cbsz:1 blgp:1\n\t" \
    "ds_read_b64_tr_b8 v[124:125], v29 offset:8192\n\t" \
    "ds_read_b64_tr_b8 v[126:127], v29 offset:9216\n\t" \
    "v_mfma_f32_16x16x128_f8f6f4 v[196:199], v[104:111], v[64:71], v[196:199] cbsz:1 blgp:1\n\t" \
    "v_add_u32 v24, %[dlt], v24\n\t" \
    "s_waitcnt lgkmcnt(12)\n\t" \
    "v_mfma_f32_16x16x128_f8f6f4 v[212:215], v[104:111], v[72:79], v[212:215] cbsz:1 blgp:1\n\t" \
    "v_add_u32 v25, %[dlt], v25\n\t" \
    "v_mfma_f32_16x16x128_f8f6f4 v[208:211], v[96:103], v[72:79], v[208:211] cbsz:1 blgp:1\n\t" \
    "v_add_u32 v26, %[dlt], v26\n\t" \
    "s_waitcnt lgkmcnt(10)\n\t" \
    "v_mfma_f32_16x16x128_f8f6f4 v[224:227], v[96:103], v[80:87], v[224:227] cbsz:1 blgp:1\n\t" \
    "v_add_u32 v27, %[dlt], v27\n\t" \
    "v_mfma_f32_16x16x128_f8f6f4 v[228:231], v[104:111], v[80:87], v[228:231] cbsz:1 blgp:1\n\t" \
    "v_add_u32 v28, %[dlt], v28\n\t" \
    "s_waitcnt lgkmcnt(8)\n\t" \
    "v_mfma_f32_16x16x128_f8f6f4 v[244:247], v[104:111], v[88:95], v[244:247] cbsz:1 blgp:1\n\t" \
    "v_add_u32 v29, %[dlt], v29\n\t" \
    "v_mfma_f32_16x16x128_f8f6f4 v[240:243], v[96:103], v[88:95], v[240:243] cbsz:1 blgp:1\n\t" \
    "L_loop_%=:\n\t" \
    "s_waitcnt lgkmcnt(0)\n\t" \
    "v_mfma_f32_16x16x128_f8f6f4 v[136:139], v[112:119], v[32:39], v[136:139] cbsz:1 blgp:1\n\t" \
    "s_barrier\n\t" \
    "ds_read_b64_tr_b8 v[96:97], v26 offset:0\n\t" \
    "ds_read_b64_tr_b8 v[98:99], v26 offset:1024\n\t" \
    "v_mfma_f32_16x16x128_f8f6f4 v[140:143], v[120:127], v[32:39], v[140:143] cbsz:1 blgp:1\n\t" \
    "ds_read_b64_tr_b8 v[100:101], v26 offset:8192\n\t" \
    "ds_read_b64_tr_b8 v[102:103], v26 offset:9216\n\t" \
    "v_mfma_f32_16x16x128_f8f6f4 v[156:159], v[120:127], v[40:47], v[156:159] cbsz:1 blgp:1\n\t" \
    "ds_read_b128 v[32:35], v24 offset:0\n\t" \
    "v_mfma_f32_16x16x128_f8f6f4 v[152:155], v[112:119], v[40:47], v[152:155] cbsz:1 blgp:1\n\t" \
    "ds_read_b128 v[36:39], v25 offset:0\n\t" \
    "v_mfma_f32_16x16x128_f8f6f4 v[168:171], v[112:119], v[48:55], v[168:171] cbsz:1 blgp:1\n\t" \
    "ds_read_b128 v[40:43], v24 offset:2048\n\t" \
    "ds_read_b64_tr_b8 v[104:105], v27 offset:0\n\t" \
    "v_mfma_f32_16x16x128_f8f6f4 v[172:175], v[120:127], v[48:55], v[172:175] cbsz:1 blgp:1\n\t" \
    "ds_read_b128 v[44:47], v25 offset:2048\n\t" \
    "ds_read_b64_tr_b8 v[106:107], v27 offset:1024\n\t" \
    "v_mfma_f32_16x16x128_f8f6f4 v[188:191], v[120:127], v[56:63], v[188:191] cbsz:1 blgp:1\n\t" \
    "ds_read_b128 v[48:51], v24 offset:4096\n\t" \
    "ds_read_b64_tr_b8 v[108:109], v27 offset:8192\n\t" \
    "v_mfma_f32_16x16x128_f8f6f4 v[184:187], v[112:119], v[56:63], v[184:187] cbsz:1 blgp:1\n\t" \
    "ds_read_b128 v[52:55], v25 offset:4096\n\t" \
    "ds_read_b64_tr_b8 v[110:111], v27 offset:9216\n\t" \
    "v_mfma_f32_16x16x128_f8f6f4 v[200:203], v[112:119], v[64:71], v[200:203] cbsz:1 blgp:1\n\t" \
    "ds_read_b128 v[56:59], v24 offset:6144\n\t" \
    "s_add_u32 %[st], %[st], 1\n\t" \
    "s_cmp_eq_u32 %[st], 3\n\t" \
    "s_cselect_b32 %[dlt], %[cneg], %[cpos]\n\t" \
    "s_cselect_b32 %[st], 0, %[st]\n\t" \
    "v_mfma_f32_16x16x128_f8f6f4 v[204:207], v[120:127], v[64:71], v[204:207] cbsz:1 blgp:1\n\t" \
    "ds_read_b128 v[60:63], v25 offset:6144\n\t" \
    "v_mfma_f32_16x16x128_f8f6f4 v[220:223], v[120:127], v[72:79], v[220:223] cbsz:1 blgp:1\n\t" \
    "ds_read_b128 v[64:67], v24 offset:8192\n\t" \
    "v_mfma_f32_16x16x128_f8f6f4 v[216:219], v[112:119], v[72:79], v[216:219] cbsz:1 blgp:1\n\t" \
    "ds_read_b128 v[68:71], v25 offset:8192\n\t" \
    "v_mfma_f32_16x16x128_f8f6f4 v[232:235], v[112:119], v[80:87], v[232:235] cbsz:1 blgp:1\n\t" \
    "ds_read_b128 v[72:75], v24 offset:10240\n\t" \
    "v_mfma_f32_16x16x128_f8f6f4 v[236:239], v[120:127], v[80:87], v[236:239] cbsz:1 blgp:1\n\t" \
    "ds_read_b128 v[76:79], v25 offset:10240\n\t" \
    "v_mfma_f32_16x16x128_f8f6f4 v[252:255], v[120:127], v[88:95], v[252:255] cbsz:1 blgp:1\n\t" \
    "ds_read_b128 v[80:83], v24 offset:12288\n\t" \
    "v_mfma_f32_16x16x128_f8f6f4 v[248:251], v[112:119], v[88:95], v[248:251] cbsz:1 blgp:1\n\t" \
    "ds_read_b128 v[84:87], v25 offset:12288\n\t" \
    "s_waitcnt lgkmcnt(15)\n\t" \
    "v_mfma_f32_16x16x128_f8f6f4 v[128:131], v[96:103], v[32:39], v[128:131] cbsz:1 blgp:1\n\t" \
    "ds_read_b128 v[88:91], v24 offset:14336\n\t" \
    "ds_read_b64_tr_b8 v[112:113], v28 offset:0\n\t" \
    "s_waitcnt lgkmcnt(10)\n\t" \
    "v_mfma_f32_16x16x128_f8f6f4 v[132:135], v[104:111], v[32:39], v[132:135] cbsz:1 blgp:1\n\t" \
    "ds_read_b128 v[92:95], v25 offset:14336\n\t" \
    "ds_read_b64_tr_b8 v[114:115], v28 offset:1024\n\t" \
    "v_mfma_f32_16x16x128_f8f6f4 v[148:151], v[104:111], v[40:47], v[148:151] cbsz:1 blgp:1\n\t" \
    "ds_read_b64_tr_b8 v[116:117], v28 offset:8192\n\t" \
    "ds_read_b64_tr_b8 v[120:121], v29 offset:0\n\t" \
    "v_mfma_f32_16x16x128_f8f6f4 v[144:147], v[96:103], v[40:47], v[144:147] cbsz:1 blgp:1\n\t" \
    "ds_read_b64_tr_b8 v[118:119], v28 offset:9216\n\t" \
    "ds_read_b64_tr_b8 v[122:123], v29 offset:1024\n\t" \
    "v_mfma_f32_16x16x128_f8f6f4 v[160:163], v[96:103], v[48:55], v[160:163] cbsz:1 blgp:1\n\t" \
    "ds_read_b64_tr_b8 v[124:125], v29 offset:8192\n\t" \
    "v_mfma_f32_16x16x128_f8f6f4 v[164:167], v[104:111], v[48:55], v[164:167] cbsz:1 blgp:1\n\t" \
    "ds_read_b64_tr_b8 v[126:127], v29 offset:9216\n\t" \
    "s_waitcnt lgkmcnt(15)\n\t" \
    "v_mfma_f32_16x16x128_f8f6f4 v[180:183], v[104:111], v[56:63], v[180:183] cbsz:1 blgp:1\n\t" \
    "v_add_u32 v24, %[dlt], v24\n\t" \
    "v_mfma_f32_16x16x128_f8f6f4 v[176:179], v[96:103], v[56:63], v[176:179] cbsz:1 blgp:1\n\t" \
    "v_add_u32 v25, %[dlt], v25\n\t" \
    "s_waitcnt lgkmcnt(14)\n\t" \
    "v_mfma_f32_16x16x128_f8f6f4 v[192:195], v[96:103], v[64:71], v[192:195] cbsz:1 blgp:1\n\t" \
    "v_add_u32 v26, %[dlt], v26\n\t" \
    "v_mfma_f32_16x16x128_f8f6f4 v[196:199], v[104:111], v[64:71], v[196:199] cbsz:1 blgp:1\n\t" \
    "v_add_u32 v27, %[dlt], v27\n\t" \
    "s_waitcnt lgkmcnt(12)\n\t" \
    "v_mfma_f32_16x16x128_f8f6f4 v[212:215], v[104:111], v[72:79], v[212:215] cbsz:1 blgp:1\n\t" \
    "v_add_u32 v28, %[dlt], v28\n\t" \
    "v_mfma_f32_16x16x128_f8f6f4 v[208:211], v[96:103], v[72:79], v[208:211] cbsz:1 blgp:1\n\t" \
    "v_add_u32 v29, %[dlt], v29\n\t" \
    "s_waitcnt lgkmcnt(10)\n\t" \
    "v_mfma_f32_16x16x128_f8f6f4 v[224:227], v[96:103], v[80:87], v[224:227] cbsz:1 blgp:1\n\t" \
    "v_mfma_f32_16x16x128_f8f6f4 v[228:231], v[104:111], v[80:87], v[228:231] cbsz:1 blgp:1\n\t" \
    "s_waitcnt lgkmcnt(7)\n\t" \
    "v_mfma_f32_16x16x128_f8f6f4 v[244:247], v[104:111], v[88:95], v[244:247] cbsz:1 blgp:1\n\t" \
    "s_sub_u32 %[cnt], %[cnt], 1\n\t" \
    "s_cmp_lg_u32 %[cnt], 0\n\t" \
    "v_mfma_f32_16x16x128_f8f6f4 v[240:243], v[96:103], v[88:95], v[240:243] cbsz:1 blgp:1\n\t" \
    "s_cbranch_scc1 L_loop_%=\n\t" \
    "ds_read_b32 v30, %[badaddr]\n\t" \
    "s_waitcnt lgkmcnt(0)\n\t" \
    "v_mfma_f32_16x16x128_f8f6f4 v[136:139], v[112:119], v[32:39], v[136:139] cbsz:1 blgp:1\n\t" \
    "v_mfma_f32_16x16x128_f8f6f4 v[140:143], v[120:127], v[32:39], v[140:143] cbsz:1 blgp:1\n\t" \
    "v_pk_mul_f32 v[128:129], v[128:129], v[30:31] op_sel_hi:[1,0]\n\t" \
    "v_pk_mul_f32 v[130:131], v[130:131], v[30:31] op_sel_hi:[1,0]\n\t" \
    "v_maximum3_f32 v128, v128, 0, 0\n\t" \
    "v_maximum3_f32 v129, v129, 0, 0\n\t" \
    "v_maximum3_f32 v130, v130, 0, 0\n\t" \
    "v_maximum3_f32 v131, v131, 0, 0\n\t" \
    "global_store_dwordx4 v16, v[128:131], %[outp] offset:0 sc1\n\t" \
    "v_mfma_f32_16x16x128_f8f6f4 v[156:159], v[120:127], v[40:47], v[156:159] cbsz:1 blgp:1\n\t" \
    "v_pk_mul_f32 v[132:133], v[132:133], v[30:31] op_sel_hi:[1,0]\n\t" \
    "v_pk_mul_f32 v[134:135], v[134:135], v[30:31] op_sel_hi:[1,0]\n\t" \
    "v_maximum3_f32 v132, v132, 0, 0\n\t" \
    "v_maximum3_f32 v133, v133, 0, 0\n\t" \
    "v_maximum3_f32 v134, v134, 0, 0\n\t" \
    "v_maximum3_f32 v135, v135, 0, 0\n\t" \
    "global_store_dwordx4 v16, v[132:135], %[outp] offset:64 sc1\n\t" \
    "v_mfma_f32_16x16x128_f8f6f4 v[152:155], v[112:119], v[40:47], v[152:155] cbsz:1 blgp:1\n\t" \
    "v_pk_mul_f32 v[144:145], v[144:145], v[30:31] op_sel_hi:[1,0]\n\t" \
    "v_pk_mul_f32 v[146:147], v[146:147], v[30:31] op_sel_hi:[1,0]\n\t" \
    "v_maximum3_f32 v144, v144, 0, 0\n\t" \
    "v_maximum3_f32 v145, v145, 0, 0\n\t" \
    "v_maximum3_f32 v146, v146, 0, 0\n\t" \
    "v_maximum3_f32 v147, v147, 0, 0\n\t" \
    "global_store_dwordx4 v17, v[144:147], %[outp] offset:0 sc1\n\t" \
    "v_mfma_f32_16x16x128_f8f6f4 v[168:171], v[112:119], v[48:55], v[168:171] cbsz:1 blgp:1\n\t" \
    "v_pk_mul_f32 v[148:149], v[148:149], v[30:31] op_sel_hi:[1,0]\n\t" \
    "v_pk_mul_f32 v[150:151], v[150:151], v[30:31] op_sel_hi:[1,0]\n\t" \
    "v_maximum3_f32 v148, v148, 0, 0\n\t" \
    "v_maximum3_f32 v149, v149, 0, 0\n\t" \
    "v_maximum3_f32 v150, v150, 0, 0\n\t" \
    "v_maximum3_f32 v151, v151, 0, 0\n\t" \
    "global_store_dwordx4 v17, v[148:151], %[outp] offset:64 sc1\n\t" \
    "v_mfma_f32_16x16x128_f8f6f4 v[172:175], v[120:127], v[48:55], v[172:175] cbsz:1 blgp:1\n\t" \
    "v_pk_mul_f32 v[160:161], v[160:161], v[30:31] op_sel_hi:[1,0]\n\t" \
    "v_pk_mul_f32 v[162:163], v[162:163], v[30:31] op_sel_hi:[1,0]\n\t" \
    "v_maximum3_f32 v160, v160, 0, 0\n\t" \
    "v_maximum3_f32 v161, v161, 0, 0\n\t" \
    "v_maximum3_f32 v162, v162, 0, 0\n\t" \
    "v_maximum3_f32 v163, v163, 0, 0\n\t" \
    "global_store_dwordx4 v18, v[160:163], %[outp] offset:0 sc1\n\t" \
    "v_mfma_f32_16x16x128_f8f6f4 v[188:191], v[120:127], v[56:63], v[188:191] cbsz:1 blgp:1\n\t" \
    "v_pk_mul_f32 v[164:165], v[164:165], v[30:31] op_sel_hi:[1,0]\n\t" \
    "v_pk_mul_f32 v[166:167], v[166:167], v[30:31] op_sel_hi:[1,0]\n\t" \
    "v_maximum3_f32 v164, v164, 0, 0\n\t" \
    "v_maximum3_f32 v165, v165, 0, 0\n\t" \
    "v_maximum3_f32 v166, v166, 0, 0\n\t" \
    "v_maximum3_f32 v167, v167, 0, 0\n\t" \
    "global_store_dwordx4 v18, v[164:167], %[outp] offset:64 sc1\n\t" \
    "v_mfma_f32_16x16x128_f8f6f4 v[184:187], v[112:119], v[56:63], v[184:187] cbsz:1 blgp:1\n\t" \
    "v_pk_mul_f32 v[176:177], v[176:177], v[30:31] op_sel_hi:[1,0]\n\t" \
    "v_pk_mul_f32 v[178:179], v[178:179], v[30:31] op_sel_hi:[1,0]\n\t" \
    "v_maximum3_f32 v176, v176, 0, 0\n\t" \
    "v_maximum3_f32 v177, v177, 0, 0\n\t" \
    "v_maximum3_f32 v178, v178, 0, 0\n\t" \
    "v_maximum3_f32 v179, v179, 0, 0\n\t" \
    "global_store_dwordx4 v19, v[176:179], %[outp] offset:0 sc1\n\t" \
    "v_mfma_f32_16x16x128_f8f6f4 v[200:203], v[112:119], v[64:71], v[200:203] cbsz:1 blgp:1\n\t" \
    "v_pk_mul_f32 v[180:181], v[180:181], v[30:31] op_sel_hi:[1,0]\n\t" \
    "v_pk_mul_f32 v[182:183], v[182:183], v[30:31] op_sel_hi:[1,0]\n\t" \
    "v_maximum3_f32 v180, v180, 0, 0\n\t" \
    "v_maximum3_f32 v181, v181, 0, 0\n\t" \
    "v_maximum3_f32 v182, v182, 0, 0\n\t" \
    "v_maximum3_f32 v183, v183, 0, 0\n\t" \
    "global_store_dwordx4 v19, v[180:183], %[outp] offset:64 sc1\n\t" \
    "v_mfma_f32_16x16x128_f8f6f4 v[204:207], v[120:127], v[64:71], v[204:207] cbsz:1 blgp:1\n\t" \
    "v_pk_mul_f32 v[192:193], v[192:193], v[30:31] op_sel_hi:[1,0]\n\t" \
    "v_pk_mul_f32 v[194:195], v[194:195], v[30:31] op_sel_hi:[1,0]\n\t" \
    "v_maximum3_f32 v192, v192, 0, 0\n\t" \
    "v_maximum3_f32 v193, v193, 0, 0\n\t" \
    "v_maximum3_f32 v194, v194, 0, 0\n\t" \
    "v_maximum3_f32 v195, v195, 0, 0\n\t" \
    "global_store_dwordx4 v20, v[192:195], %[outp] offset:0 sc1\n\t" \
    "v_mfma_f32_16x16x128_f8f6f4 v[220:223], v[120:127], v[72:79], v[220:223] cbsz:1 blgp:1\n\t" \
    "v_pk_mul_f32 v[196:197], v[196:197], v[30:31] op_sel_hi:[1,0]\n\t" \
    "v_pk_mul_f32 v[198:199], v[198:199], v[30:31] op_sel_hi:[1,0]\n\t" \
    "v_maximum3_f32 v196, v196, 0, 0\n\t" \
    "v_maximum3_f32 v197, v197, 0, 0\n\t" \
    "v_maximum3_f32 v198, v198, 0, 0\n\t" \
    "v_maximum3_f32 v199, v199, 0, 0\n\t" \
    "global_store_dwordx4 v20, v[196:199], %[outp] offset:64 sc1\n\t" \
    "v_mfma_f32_16x16x128_f8f6f4 v[216:219], v[112:119], v[72:79], v[216:219] cbsz:1 blgp:1\n\t" \
    "v_pk_mul_f32 v[208:209], v[208:209], v[30:31] op_sel_hi:[1,0]\n\t" \
    "v_pk_mul_f32 v[210:211], v[210:211], v[30:31] op_sel_hi:[1,0]\n\t" \
    "v_maximum3_f32 v208, v208, 0, 0\n\t" \
    "v_maximum3_f32 v209, v209, 0, 0\n\t" \
    "v_maximum3_f32 v210, v210, 0, 0\n\t" \
    "v_maximum3_f32 v211, v211, 0, 0\n\t" \
    "global_store_dwordx4 v21, v[208:211], %[outp] offset:0 sc1\n\t" \
    "v_mfma_f32_16x16x128_f8f6f4 v[232:235], v[112:119], v[80:87], v[232:235] cbsz:1 blgp:1\n\t" \
    "v_pk_mul_f32 v[212:213], v[212:213], v[30:31] op_sel_hi:[1,0]\n\t" \
    "v_pk_mul_f32 v[214:215], v[214:215], v[30:31] op_sel_hi:[1,0]\n\t" \
    "v_maximum3_f32 v212, v212, 0, 0\n\t" \
    "v_maximum3_f32 v213, v213, 0, 0\n\t" \
    "v_maximum3_f32 v214, v214, 0, 0\n\t" \
    "v_maximum3_f32 v215, v215, 0, 0\n\t" \
    "global_store_dwordx4 v21, v[212:215], %[outp] offset:64 sc1\n\t" \
    "v_mfma_f32_16x16x128_f8f6f4 v[236:239], v[120:127], v[80:87], v[236:239] cbsz:1 blgp:1\n\t" \
    "v_pk_mul_f32 v[224:225], v[224:225], v[30:31] op_sel_hi:[1,0]\n\t" \
    "v_pk_mul_f32 v[226:227], v[226:227], v[30:31] op_sel_hi:[1,0]\n\t" \
    "v_maximum3_f32 v224, v224, 0, 0\n\t" \
    "v_maximum3_f32 v225, v225, 0, 0\n\t" \
    "v_maximum3_f32 v226, v226, 0, 0\n\t" \
    "v_maximum3_f32 v227, v227, 0, 0\n\t" \
    "global_store_dwordx4 v22, v[224:227], %[outp] offset:0 sc1\n\t" \
    "v_mfma_f32_16x16x128_f8f6f4 v[252:255], v[120:127], v[88:95], v[252:255] cbsz:1 blgp:1\n\t" \
    "v_pk_mul_f32 v[228:229], v[228:229], v[30:31] op_sel_hi:[1,0]\n\t" \
    "v_pk_mul_f32 v[230:231], v[230:231], v[30:31] op_sel_hi:[1,0]\n\t" \
    "v_maximum3_f32 v228, v228, 0, 0\n\t" \
    "v_maximum3_f32 v229, v229, 0, 0\n\t" \
    "v_maximum3_f32 v230, v230, 0, 0\n\t" \
    "v_maximum3_f32 v231, v231, 0, 0\n\t" \
    "global_store_dwordx4 v22, v[228:231], %[outp] offset:64 sc1\n\t" \
    "v_mfma_f32_16x16x128_f8f6f4 v[248:251], v[112:119], v[88:95], v[248:251] cbsz:1 blgp:1\n\t" \
    "v_pk_mul_f32 v[240:241], v[240:241], v[30:31] op_sel_hi:[1,0]\n\t" \
    "v_pk_mul_f32 v[242:243], v[242:243], v[30:31] op_sel_hi:[1,0]\n\t" \
    "v_maximum3_f32 v240, v240, 0, 0\n\t" \
    "v_maximum3_f32 v241, v241, 0, 0\n\t" \
    "v_maximum3_f32 v242, v242, 0, 0\n\t" \
    "v_maximum3_f32 v243, v243, 0, 0\n\t" \
    "global_store_dwordx4 v23, v[240:243], %[outp] offset:0 sc1\n\t" \
    "v_pk_mul_f32 v[244:245], v[244:245], v[30:31] op_sel_hi:[1,0]\n\t" \
    "v_pk_mul_f32 v[246:247], v[246:247], v[30:31] op_sel_hi:[1,0]\n\t" \
    "v_maximum3_f32 v244, v244, 0, 0\n\t" \
    "v_maximum3_f32 v245, v245, 0, 0\n\t" \
    "v_maximum3_f32 v246, v246, 0, 0\n\t" \
    "v_maximum3_f32 v247, v247, 0, 0\n\t" \
    "global_store_dwordx4 v23, v[244:247], %[outp] offset:64 sc1\n\t" \
    "v_pk_mul_f32 v[136:137], v[136:137], v[30:31] op_sel_hi:[1,0]\n\t" \
    "v_pk_mul_f32 v[138:139], v[138:139], v[30:31] op_sel_hi:[1,0]\n\t" \
    "v_maximum3_f32 v136, v136, 0, 0\n\t" \
    "v_maximum3_f32 v137, v137, 0, 0\n\t" \
    "v_maximum3_f32 v138, v138, 0, 0\n\t" \
    "v_maximum3_f32 v139, v139, 0, 0\n\t" \
    "global_store_dwordx4 v16, v[136:139], %[outp] offset:128 sc1\n\t" \
    "v_pk_mul_f32 v[140:141], v[140:141], v[30:31] op_sel_hi:[1,0]\n\t" \
    "v_pk_mul_f32 v[142:143], v[142:143], v[30:31] op_sel_hi:[1,0]\n\t" \
    "v_maximum3_f32 v140, v140, 0, 0\n\t" \
    "v_maximum3_f32 v141, v141, 0, 0\n\t" \
    "v_maximum3_f32 v142, v142, 0, 0\n\t" \
    "v_maximum3_f32 v143, v143, 0, 0\n\t" \
    "global_store_dwordx4 v16, v[140:143], %[outp] offset:192 sc1\n\t" \
    "v_pk_mul_f32 v[152:153], v[152:153], v[30:31] op_sel_hi:[1,0]\n\t" \
    "v_pk_mul_f32 v[154:155], v[154:155], v[30:31] op_sel_hi:[1,0]\n\t" \
    "v_maximum3_f32 v152, v152, 0, 0\n\t" \
    "v_maximum3_f32 v153, v153, 0, 0\n\t" \
    "v_maximum3_f32 v154, v154, 0, 0\n\t" \
    "v_maximum3_f32 v155, v155, 0, 0\n\t" \
    "global_store_dwordx4 v17, v[152:155], %[outp] offset:128 sc1\n\t" \
    "v_pk_mul_f32 v[156:157], v[156:157], v[30:31] op_sel_hi:[1,0]\n\t" \
    "v_pk_mul_f32 v[158:159], v[158:159], v[30:31] op_sel_hi:[1,0]\n\t" \
    "v_maximum3_f32 v156, v156, 0, 0\n\t" \
    "v_maximum3_f32 v157, v157, 0, 0\n\t" \
    "v_maximum3_f32 v158, v158, 0, 0\n\t" \
    "v_maximum3_f32 v159, v159, 0, 0\n\t" \
    "global_store_dwordx4 v17, v[156:159], %[outp] offset:192 sc1\n\t" \
    "v_pk_mul_f32 v[168:169], v[168:169], v[30:31] op_sel_hi:[1,0]\n\t" \
    "v_pk_mul_f32 v[170:171], v[170:171], v[30:31] op_sel_hi:[1,0]\n\t" \
    "v_maximum3_f32 v168, v168, 0, 0\n\t" \
    "v_maximum3_f32 v169, v169, 0, 0\n\t" \
    "v_maximum3_f32 v170, v170, 0, 0\n\t" \
    "v_maximum3_f32 v171, v171, 0, 0\n\t" \
    "global_store_dwordx4 v18, v[168:171], %[outp] offset:128 sc1\n\t" \
    "v_pk_mul_f32 v[172:173], v[172:173], v[30:31] op_sel_hi:[1,0]\n\t" \
    "v_pk_mul_f32 v[174:175], v[174:175], v[30:31] op_sel_hi:[1,0]\n\t" \
    "v_maximum3_f32 v172, v172, 0, 0\n\t" \
    "v_maximum3_f32 v173, v173, 0, 0\n\t" \
    "v_maximum3_f32 v174, v174, 0, 0\n\t" \
    "v_maximum3_f32 v175, v175, 0, 0\n\t" \
    "global_store_dwordx4 v18, v[172:175], %[outp] offset:192 sc1\n\t" \
    "v_pk_mul_f32 v[184:185], v[184:185], v[30:31] op_sel_hi:[1,0]\n\t" \
    "v_pk_mul_f32 v[186:187], v[186:187], v[30:31] op_sel_hi:[1,0]\n\t" \
    "v_maximum3_f32 v184, v184, 0, 0\n\t" \
    "v_maximum3_f32 v185, v185, 0, 0\n\t" \
    "v_maximum3_f32 v186, v186, 0, 0\n\t" \
    "v_maximum3_f32 v187, v187, 0, 0\n\t" \
    "global_store_dwordx4 v19, v[184:187], %[outp] offset:128 sc1\n\t" \
    "v_pk_mul_f32 v[188:189], v[188:189], v[30:31] op_sel_hi:[1,0]\n\t" \
    "v_pk_mul_f32 v[190:191], v[190:191], v[30:31] op_sel_hi:[1,0]\n\t" \
    "v_maximum3_f32 v188, v188, 0, 0\n\t" \
    "v_maximum3_f32 v189, v189, 0, 0\n\t" \
    "v_maximum3_f32 v190, v190, 0, 0\n\t" \
    "v_maximum3_f32 v191, v191, 0, 0\n\t" \
    "global_store_dwordx4 v19, v[188:191], %[outp] offset:192 sc1\n\t" \
    "v_pk_mul_f32 v[200:201], v[200:201], v[30:31] op_sel_hi:[1,0]\n\t" \
    "v_pk_mul_f32 v[202:203], v[202:203], v[30:31] op_sel_hi:[1,0]\n\t" \
    "v_maximum3_f32 v200, v200, 0, 0\n\t" \
    "v_maximum3_f32 v201, v201, 0, 0\n\t" \
    "v_maximum3_f32 v202, v202, 0, 0\n\t" \
    "v_maximum3_f32 v203, v203, 0, 0\n\t" \
    "global_store_dwordx4 v20, v[200:203], %[outp] offset:128 sc1\n\t" \
    "v_pk_mul_f32 v[204:205], v[204:205], v[30:31] op_sel_hi:[1,0]\n\t" \
    "v_pk_mul_f32 v[206:207], v[206:207], v[30:31] op_sel_hi:[1,0]\n\t" \
    "v_maximum3_f32 v204, v204, 0, 0\n\t" \
    "v_maximum3_f32 v205, v205, 0, 0\n\t" \
    "v_maximum3_f32 v206, v206, 0, 0\n\t" \
    "v_maximum3_f32 v207, v207, 0, 0\n\t" \
    "global_store_dwordx4 v20, v[204:207], %[outp] offset:192 sc1\n\t" \
    "v_pk_mul_f32 v[216:217], v[216:217], v[30:31] op_sel_hi:[1,0]\n\t" \
    "v_pk_mul_f32 v[218:219], v[218:219], v[30:31] op_sel_hi:[1,0]\n\t" \
    "v_maximum3_f32 v216, v216, 0, 0\n\t" \
    "v_maximum3_f32 v217, v217, 0, 0\n\t" \
    "v_maximum3_f32 v218, v218, 0, 0\n\t" \
    "v_maximum3_f32 v219, v219, 0, 0\n\t" \
    "global_store_dwordx4 v21, v[216:219], %[outp] offset:128 sc1\n\t" \
    "v_pk_mul_f32 v[220:221], v[220:221], v[30:31] op_sel_hi:[1,0]\n\t" \
    "v_pk_mul_f32 v[222:223], v[222:223], v[30:31] op_sel_hi:[1,0]\n\t" \
    "v_maximum3_f32 v220, v220, 0, 0\n\t" \
    "v_maximum3_f32 v221, v221, 0, 0\n\t" \
    "v_maximum3_f32 v222, v222, 0, 0\n\t" \
    "v_maximum3_f32 v223, v223, 0, 0\n\t" \
    "global_store_dwordx4 v21, v[220:223], %[outp] offset:192 sc1\n\t" \
    "v_pk_mul_f32 v[232:233], v[232:233], v[30:31] op_sel_hi:[1,0]\n\t" \
    "v_pk_mul_f32 v[234:235], v[234:235], v[30:31] op_sel_hi:[1,0]\n\t" \
    "v_maximum3_f32 v232, v232, 0, 0\n\t" \
    "v_maximum3_f32 v233, v233, 0, 0\n\t" \
    "v_maximum3_f32 v234, v234, 0, 0\n\t" \
    "v_maximum3_f32 v235, v235, 0, 0\n\t" \
    "global_store_dwordx4 v22, v[232:235], %[outp] offset:128 sc1\n\t" \
    "v_pk_mul_f32 v[236:237], v[236:237], v[30:31] op_sel_hi:[1,0]\n\t" \
    "v_pk_mul_f32 v[238:239], v[238:239], v[30:31] op_sel_hi:[1,0]\n\t" \
    "v_maximum3_f32 v236, v236, 0, 0\n\t" \
    "v_maximum3_f32 v237, v237, 0, 0\n\t" \
    "v_maximum3_f32 v238, v238, 0, 0\n\t" \
    "v_maximum3_f32 v239, v239, 0, 0\n\t" \
    "global_store_dwordx4 v22, v[236:239], %[outp] offset:192 sc1\n\t" \
    "v_pk_mul_f32 v[248:249], v[248:249], v[30:31] op_sel_hi:[1,0]\n\t" \
    "v_pk_mul_f32 v[250:251], v[250:251], v[30:31] op_sel_hi:[1,0]\n\t" \
    "v_maximum3_f32 v248, v248, 0, 0\n\t" \
    "v_maximum3_f32 v249, v249, 0, 0\n\t" \
    "v_maximum3_f32 v250, v250, 0, 0\n\t" \
    "v_maximum3_f32 v251, v251, 0, 0\n\t" \
    "global_store_dwordx4 v23, v[248:251], %[outp] offset:128 sc1\n\t" \
    "v_pk_mul_f32 v[252:253], v[252:253], v[30:31] op_sel_hi:[1,0]\n\t" \
    "v_pk_mul_f32 v[254:255], v[254:255], v[30:31] op_sel_hi:[1,0]\n\t" \
    "v_maximum3_f32 v252, v252, 0, 0\n\t" \
    "v_maximum3_f32 v253, v253, 0, 0\n\t" \
    "v_maximum3_f32 v254, v254, 0, 0\n\t" \
    "v_maximum3_f32 v255, v255, 0, 0\n\t" \
    "global_store_dwordx4 v23, v[252:255], %[outp] offset:192 sc1\n\t"
#define COMPUTE_CLOBBERS "v16", "v17", "v18", "v19", "v20", "v21", "v22", "v23", "v24", "v25", "v26", "v27", "v28", "v29", "v30", "v31", "v32", "v33", "v34", "v35", "v36", "v37", "v38", "v39", "v40", "v41", "v42", "v43", "v44", "v45", "v46", "v47", "v48", "v49", "v50", "v51", "v52", "v53", "v54", "v55", "v56", "v57", "v58", "v59", "v60", "v61", "v62", "v63", "v64", "v65", "v66", "v67", "v68", "v69", "v70", "v71", "v72", "v73", "v74", "v75", "v76", "v77", "v78", "v79", "v80", "v81", "v82", "v83", "v84", "v85", "v86", "v87", "v88", "v89", "v90", "v91", "v92", "v93", "v94", "v95", "v96", "v97", "v98", "v99", "v100", "v101", "v102", "v103", "v104", "v105", "v106", "v107", "v108", "v109", "v110", "v111", "v112", "v113", "v114", "v115", "v116", "v117", "v118", "v119", "v120", "v121", "v122", "v123", "v124", "v125", "v126", "v127", "v128", "v129", "v130", "v131", "v132", "v133", "v134", "v135", "v136", "v137", "v138", "v139", "v140", "v141", "v142", "v143", "v144", "v145", "v146", "v147", "v148", "v149", "v150", "v151", "v152", "v153", "v154", "v155", "v156", "v157", "v158", "v159", "v160", "v161", "v162", "v163", "v164", "v165", "v166", "v167", "v168", "v169", "v170", "v171", "v172", "v173", "v174", "v175", "v176", "v177", "v178", "v179", "v180", "v181", "v182", "v183", "v184", "v185", "v186", "v187", "v188", "v189", "v190", "v191", "v192", "v193", "v194", "v195", "v196", "v197", "v198", "v199", "v200", "v201", "v202", "v203", "v204", "v205", "v206", "v207", "v208", "v209", "v210", "v211", "v212", "v213", "v214", "v215", "v216", "v217", "v218", "v219", "v220", "v221", "v222", "v223", "v224", "v225", "v226", "v227", "v228", "v229", "v230", "v231", "v232", "v233", "v234", "v235", "v236", "v237", "v238", "v239", "v240", "v241", "v242", "v243", "v244", "v245", "v246", "v247", "v248", "v249", "v250", "v251", "v252", "v253", "v254", "v255"
#define NBAR 1
#include <hip/hip_runtime.h>
#include <cstdio>
#include <cstdint>

namespace a2a {
constexpr int NR = 8;
constexpr int MSH = 512, KSH = 512;
constexpr int KTOT = 4096, NTOT = 16384;
constexpr int NSTEP = 32;
constexpr unsigned STAGE = 49152, LDS_B_OFF = 32768;
constexpr unsigned LDS_POLL = 3 * STAGE;
constexpr unsigned LDS_ORD = LDS_POLL + 512;
constexpr unsigned LDS_PF = LDS_ORD + 128;
constexpr unsigned LDS_SCALE = LDS_PF + 256;
constexpr unsigned LDS_BYTES = LDS_SCALE + 64;
constexpr size_t WS_CNT = 0;
constexpr size_t WS_TMO = 32768;
constexpr size_t WS_FLAG = 65536;
constexpr size_t WS_DATA = 1u << 20;
constexpr size_t WS_NEED = WS_DATA + 2u * NR * MSH * KSH;
constexpr unsigned long long DEADLINE_TICKS = 40000000ull;
constexpr int PUSH_B1 = 3, PUSH_B2 = 6, PUSH_B3 = 9;
constexpr int PF_AHEAD = 7, PF_PER = 4, PF_PRE = 3;
constexpr int POLL_LOW = 3;

struct Args {
  const unsigned char* x;
  const unsigned char* w;
  float* out;
  unsigned char* myws;
  const float* sx; const float* sw;
  int rank; int pad;
  unsigned long long* dbg;
  unsigned char* peer[NR - 1];
};

typedef unsigned u32x4 __attribute__((ext_vector_type(4)));
#define DI __device__ __forceinline__
DI unsigned rfl(unsigned v) { return __builtin_amdgcn_readfirstlane(v); }

#define AMOD " sc0 sc1"
#define DMA1(V, MOD) "s_add_u32 m0, m0, 0x400\n\ts_add_u32 %[so], %[so], %[sd]\n\tbuffer_load_dwordx4 " V ", %[rs], %[so] offen" MOD " lds\n\t"
#define DMA0(V, MOD) "s_mov_b32 m0, %[m0v]\n\ts_nop 0\n\tbuffer_load_dwordx4 " V ", %[rs], %[so] offen" MOD " lds\n\t"
#define DMA_EO(MOD) DMA1("%[ve]", MOD) DMA1("%[vo]", MOD)
#define DMA_OPS : [so] "+s"(so) : [rs] "s"(rs), [m0v] "s"(m0v), [sd] "s"(sd), [ve] "v"(ve), [vo] "v"(vo) : "memory", "scc"
DI void dmaA8(__amdgpu_buffer_rsrc_t rs, unsigned m0v, unsigned so, unsigned sd, unsigned ve, unsigned vo) {
  asm volatile(DMA0("%[ve]", AMOD) DMA1("%[vo]", AMOD) DMA_EO(AMOD) DMA_EO(AMOD) DMA_EO(AMOD) DMA_OPS);
}
DI void dmaA24(__amdgpu_buffer_rsrc_t rs, unsigned m0v, unsigned so, unsigned sd, unsigned ve, unsigned vo) {
  asm volatile(DMA0("%[ve]", AMOD) DMA1("%[vo]", AMOD) DMA_EO(AMOD) DMA_EO(AMOD) DMA_EO(AMOD) DMA_EO(AMOD) DMA_EO(AMOD)
               DMA_EO(AMOD) DMA_EO(AMOD) DMA_EO(AMOD) DMA_EO(AMOD) DMA_EO(AMOD) DMA_EO(AMOD) DMA_OPS);
}
DI void dmaB16(__amdgpu_buffer_rsrc_t rs, unsigned m0v, unsigned so, unsigned sd, unsigned ve, unsigned vo) {
  asm volatile(DMA0("%[ve]", "") DMA1("%[ve]", "") DMA1("%[vo]", "") DMA1("%[vo]", "") DMA1("%[ve]", "") DMA1("%[ve]", "") DMA1("%[vo]", "") DMA1("%[vo]", "")
               DMA1("%[ve]", "") DMA1("%[ve]", "") DMA1("%[vo]", "") DMA1("%[vo]", "") DMA1("%[ve]", "") DMA1("%[ve]", "") DMA1("%[vo]", "") DMA1("%[vo]", "") DMA_OPS);
}
DI unsigned pos_id(unsigned p, unsigned rank) {
  if (p < 4u) return rank * 4u + p;
  const unsigned q = p - 4u, kk = q / 7u;
  return (((rank + 1u + (q - kk * 7u)) & 7u) << 2) + kk;
}
DI unsigned vmcnt_now() {
  const unsigned v = __builtin_amdgcn_s_getreg((31 << 11) | 7);
  return (v & 15u) | ((v >> 18) & 48u);
}
DI unsigned load_call(const unsigned* cntp) {
  unsigned c;
  asm volatile("s_load_dword %0, %1, 0x0\n\ts_waitcnt lgkmcnt(0)" : "=s"(c) : "s"(cntp) : "memory");
  return c + 1u;
}

__global__ __launch_bounds__(512, 2) void a2a_gemm(Args a) {
  extern __shared__ __attribute__((aligned(128))) unsigned char lds[];
  const unsigned lane = threadIdx.x & 63u;
  const unsigned wid = rfl(threadIdx.x >> 6);
  const unsigned bid = blockIdx.x;
  const unsigned xcd = bid & 7u, idx = bid >> 3;
  const unsigned pr = idx >> 1;
  const unsigned tp = pr == 3u ? 14u : pr == 14u ? 3u : pr == 11u ? 15u : pr == 15u ? 11u : pr;
  const unsigned ntile = xcd * 16u + tp, mhalf = idx & 1u;
  const unsigned ldsb = (unsigned)(size_t)lds;
  const unsigned rank = (unsigned)a.rank & 7u;
  unsigned char* myws = a.myws;
  unsigned* cntp = (unsigned*)(myws + WS_CNT + bid * 128u);

  if (wid < 4) {
    const unsigned wm = wid & 1u, wn = wid >> 1;
    const unsigned g = lane >> 4, r16 = lane & 15u, q = r16 >> 1, p = lane & 1u;
    const unsigned xa = ldsb + wm * 16384u + r16 * 128u + 16u * (g ^ q);
    const unsigned f = (q >> 1) | ((g & 1u) << 2);
    const unsigned wb = ldsb + LDS_B_OFF + (16u * g + q) * 128u + 16u * ((wn * 4u) ^ f) + 8u * p;
    const unsigned voff = ((mhalf * 256u + wm * 128u + r16) * (unsigned)NTOT + ntile * 128u + wn * 64u + 4u * g) * 4u;
    const unsigned badaddr = ldsb + LDS_SCALE;
    unsigned st, cnt, cpos, cneg, dlt;
    asm volatile(COMPUTE_ASM
                 : [st] "=&s"(st), [cnt] "=&s"(cnt), [cpos] "=&s"(cpos), [cneg] "=&s"(cneg), [dlt] "=&s"(dlt)
                 : [xa] "v"(xa), [wb] "v"(wb), [voff] "v"(voff), [badaddr] "v"(badaddr), [outp] "s"(a.out)
                 : "memory", "scc", "vcc", COMPUTE_CLOBBERS);
  } else if (wid < 6) {
    const bool second = wid == 5u;
    const unsigned c0 = (lane & 7u) ^ (lane >> 4);
    const unsigned ve = (lane >> 3) * 512u + 16u * c0, vo = (lane >> 3) * 512u + 16u * (c0 ^ 4u);
    const unsigned we = (lane >> 3) * (unsigned)NTOT + 16u * c0, wo = (lane >> 3) * (unsigned)NTOT + 16u * (c0 ^ 4u);
    const unsigned rowoff = (mhalf * 256u + (second ? 192u : 0u)) * 512u;
    const unsigned am0 = ldsb + (second ? 24576u : 0u), bm0 = ldsb + LDS_B_OFF, n0 = ntile * 128u;
    const __amdgpu_buffer_rsrc_t rsL = __builtin_amdgcn_make_buffer_rsrc((void*)(a.x + (size_t)rank * (MSH * KSH)), 0, MSH * KSH, 0x00020000);
    const __amdgpu_buffer_rsrc_t rsW = __builtin_amdgcn_make_buffer_rsrc((void*)a.w, 0, KTOT * NTOT, 0x00020000);
    if (second) {
      dmaB16(rsW, bm0, (rank * 512u) * NTOT + n0, 8u * NTOT, we, wo);
      dmaA8(rsL, am0, rowoff, 4096u, ve, vo);
      dmaB16(rsW, bm0 + STAGE, (rank * 512u + 128u) * NTOT + n0, 8u * NTOT, we, wo);
      dmaA8(rsL, am0 + STAGE, rowoff + 128u, 4096u, ve, vo);
    } else {
      dmaA24(rsL, am0, rowoff, 4096u, ve, vo);
      dmaA24(rsL, am0 + STAGE, rowoff + 128u, 4096u, ve, vo);
    }
    const unsigned call = load_call(cntp);
    const __amdgpu_buffer_rsrc_t rsR = __builtin_amdgcn_make_buffer_rsrc((void*)(myws + WS_DATA + (size_t)(call & 1u) * (NR * MSH * KSH)), 0, NR * MSH * KSH, 0x00020000);
    unsigned slot = 2;
    for (int s = 0; s < NSTEP; ++s) {
      if (s < NSTEP - 1) asm volatile("s_waitcnt vmcnt(24)\n\ts_barrier" ::: "memory");
      else               asm volatile("s_waitcnt vmcnt(0)\n\ts_barrier" ::: "memory");
      if (s + 2 < NSTEP) {
        unsigned id = rank * 4u + 2u;
        if (s >= 1) {
          unsigned idv;
          asm volatile("ds_read_b32 %0, %1\n\ts_waitcnt lgkmcnt(0)" : "=v"(idv) : "v"(ldsb + LDS_ORD + (unsigned)(s + 2) * 4u) : "memory");
          id = rfl(idv) & 31u;
        }
        const unsigned src = id >> 2, kk = id & 3u, so = slot * STAGE;
        if (second) dmaB16(rsW, bm0 + so, (id * 128u) * NTOT + n0, 8u * NTOT, we, wo);
        if (src == rank) { if (second) dmaA8(rsL, am0 + so, rowoff + kk * 128u, 4096u, ve, vo); else dmaA24(rsL, am0 + so, rowoff + kk * 128u, 4096u, ve, vo); }
        else { const unsigned o = src * (MSH * KSH) + rowoff + kk * 128u;
               if (second) dmaA8(rsR, am0 + so, o, 4096u, ve, vo); else dmaA24(rsR, am0 + so, o, 4096u, ve, vo); }
        slot = slot == 2 ? 0 : slot + 1;
      }
    }
  } else if (wid == 6) {
    const float sxv = a.sx[0], swv = a.sw[0];
    const unsigned call = load_call(cntp);
    const u32x4* flagp = (const u32x4*)(myws + WS_FLAG + mhalf * 512u) + (lane & 31u);
    unsigned* tmo = (unsigned*)(myws + WS_TMO);
    unsigned ready = 0xfu;
    unsigned taken = 0x7u;
    bool snap = false;
    for (int s = 0; s < NSTEP; ++s) {
      if (s + 2 < NSTEP) {
        u32x4 pv;
        if (snap) {
          asm volatile("s_waitcnt vmcnt(0)\n\tds_read_b128 %0, %1\n\ts_waitcnt lgkmcnt(0)" : "=v"(pv) : "v"(ldsb + LDS_POLL + (lane & 31u) * 16u) : "memory");
          ready |= (unsigned)__ballot(pv.x >= call && pv.y >= call && pv.z >= call && pv.w >= call);
          snap = false;
        }
        if (s >= 1) {
          if ((ready & ~taken) == 0u) {
            const unsigned long long t0 = __builtin_amdgcn_s_memrealtime();
            for (unsigned it = 1;; ++it) {
              asm volatile("global_load_dwordx4 %0, %1, off sc0 sc1\n\ts_waitcnt vmcnt(0)" : "=v"(pv) : "v"(flagp) : "memory");
              ready |= (unsigned)__ballot(pv.x >= call && pv.y >= call && pv.z >= call && pv.w >= call);
              if (ready & ~taken) break;
              if ((it & 31u) == 0u &&
                  (rfl(__hip_atomic_load(tmo, __ATOMIC_RELAXED, __HIP_MEMORY_SCOPE_AGENT)) == call ||
                   __builtin_amdgcn_s_memrealtime() - t0 > DEADLINE_TICKS)) {
                if (lane == 0) __hip_atomic_store(tmo, call, __ATOMIC_RELAXED, __HIP_MEMORY_SCOPE_AGENT);
                asm volatile("ds_write_b32 %0, %1\n\ts_waitcnt vmcnt(0)" :: "v"(ldsb + LDS_SCALE), "v"(0x7fc00000u) : "memory");
                ready = ~0u;
                break;
              }
              __builtin_amdgcn_s_sleep(2);
            }
          }
          const unsigned p = (unsigned)__builtin_ctz(ready & ~taken);
          taken |= 1u << p;
          asm volatile("ds_write_b32 %0, %1" :: "v"(ldsb + LDS_ORD + (unsigned)(s + 2) * 4u), "v"(pos_id(p, rank)) : "memory");
        }
        if (ready != ~0u && __builtin_popcount(ready & ~taken) < POLL_LOW) {
          if (lane < 32u) asm volatile("s_mov_b32 m0, %0\n\ts_nop 0\n\tglobal_load_lds_dwordx4 %1, off sc0 sc1" :: "s"(ldsb + LDS_POLL), "v"(flagp) : "memory");
          snap = true;
        }
      }
      asm volatile("s_waitcnt lgkmcnt(0)\n\ts_barrier" ::: "memory");
      if (s == 0) {
        asm volatile("ds_write_b32 %0, %1" :: "v"(ldsb + LDS_SCALE), "v"(sxv * swv) : "memory");
        if (lane == 0) __hip_atomic_store(cntp, call, __ATOMIC_RELAXED, __HIP_MEMORY_SCOPE_AGENT);
      }
    }
  } else {
    const bool pushes = bid < 224u;
    const unsigned job = pushes ? bid : 0u;
    const unsigned kk = job / 56u, r = job - kk * 56u, rb = r / 7u, dd = r - rb * 7u;
    const unsigned dest = (rank + 1u + dd) & 7u;
    const unsigned lo = (lane >> 3) * 512u + (lane & 7u) * 16u + rb * (64u * 512u) + kk * 128u;
    const unsigned char* srcp = a.x + (size_t)dest * (MSH * KSH) + lo;
    unsigned char* pw = a.peer[dd];
    u32x4 d[8];
#pragma unroll
    for (int i = 0; i < 8; ++i) d[i] = *(const u32x4*)(srcp + i * 4096);
    asm volatile("" : "+s"(pw) :: "memory");
    const unsigned call = load_call(cntp);
    unsigned char* dstp = pw + WS_DATA + (size_t)(call & 1u) * (NR * MSH * KSH) + (size_t)rank * (MSH * KSH) + lo;
    unsigned* fl = (unsigned*)(pw + WS_FLAG + (rb >> 2) * 512u) + (4u + kk * 7u + 6u - dd) * 4u + (rb & 3u);
    const unsigned char* pfp = a.w + (size_t)(idx * 4u + (lane >> 4)) * NTOT + xcd * 2048u + (lane & 15u) * 128u;
    unsigned pf = 2;
#define TOUCH_W() do { const unsigned char* pp = pfp + (size_t)pos_id(pf, rank) * (128u * NTOT); \
      asm volatile("s_mov_b32 m0, %0\n\ts_nop 0\n\tglobal_load_lds_dword %1, off" :: "s"(ldsb + LDS_PF), "v"(pp) : "memory"); ++pf; } while (0)
    asm volatile("s_waitcnt vmcnt(0)" ::: "memory");
    const int bstart = kk == 0u ? 0 : kk == 1u ? PUSH_B1 : kk == 2u ? PUSH_B2 : PUSH_B3, bforce = 2 + 7 * (int)kk;
    unsigned pst = pushes ? 0u : 2u;
    for (int b = 0; b < NSTEP; ++b) {
      if (pst == 0u && b >= bstart) {
        if (b > 0) for (int t = 0; t < PF_PRE && pf < (unsigned)NSTEP; ++t) TOUCH_W();
#pragma unroll
        for (int i = 0; i < 8; ++i)
          asm volatile("global_store_dwordx4 %0, %1, off sc0 sc1\n\ts_nop 1" :: "v"(dstp + i * 4096), "v"(d[i]) : "memory");
        pst = 1u;
      } else if (pst == 1u) {
        bool done = vmcnt_now() == 0u;
        for (int t = 0; t < 2 && !done; ++t) { __builtin_amdgcn_s_sleep(1); done = vmcnt_now() == 0u; }
        if (done || b >= bforce) {
          asm volatile("s_waitcnt vmcnt(0)" ::: "memory");
          if (lane == 0) asm volatile("global_store_dword %0, %1, off sc0 sc1" :: "v"(fl), "v"(call) : "memory");
          pst = 2u;
        }
      }
      if (pst != 1u)
        for (int t = 0; t < PF_PER && pf < (unsigned)NSTEP && pf < (unsigned)(b + PF_AHEAD) && vmcnt_now() < 40u; ++t) TOUCH_W();
      if (pst == 2u && pf >= (unsigned)NSTEP) return;
      asm volatile("s_barrier" ::: "memory");
    }
  }
}
}

#include <rccl/rccl.h>
#define HIP_CHECK(expr) do { hipError_t _e = (expr); if (_e != hipSuccess) { \
    fprintf(stderr, "HIP error %s at %s:%d\n", hipGetErrorString(_e), __FILE__, __LINE__); \
    return; } } while (0)

extern "C" void kernel_launch(int rank, int nranks, ncclComm_t comm,
                              hipStream_t stream,
                              void* const* d_in, const int* in_sizes, int n_in,
                              void* d_out, int out_size,
                              const int* in_dtypes, int out_dtype,
                              void* const* d_out_all,
                              void* const* d_ws_all, size_t ws_bytes) {
    using namespace a2a;
    if (nranks != NR || (unsigned)rank >= (unsigned)NR || n_in != 4 || ws_bytes < WS_NEED ||
        in_sizes[0] != NR * MSH * KSH || in_sizes[1] != KTOT * NTOT || out_size != MSH * NTOT) {
        fprintf(stderr, "kernel_launch: unexpected problem shape\n");
        return;
    }
    static bool attr_set = false;
    if (!attr_set) {
        HIP_CHECK(hipFuncSetAttribute((const void*)a2a_gemm, hipFuncAttributeMaxDynamicSharedMemorySize, LDS_BYTES));
        attr_set = true;
    }
    Args a;
    a.x = (const unsigned char*)d_in[0];
    a.w = (const unsigned char*)d_in[1];
    a.sx = (const float*)d_in[2];
    a.sw = (const float*)d_in[3];
    a.out = (float*)d_out;
    a.myws = (unsigned char*)d_ws_all[rank];
    for (int i = 0; i < NR - 1; ++i) a.peer[i] = (unsigned char*)d_ws_all[(rank + 1 + i) % NR];
    a.rank = rank; a.pad = 0;
    a.dbg = nullptr;
    hipLaunchKernelGGL(a2a_gemm, dim3(256), dim3(512), LDS_BYTES, stream, a);
    HIP_CHECK(hipGetLastError());
}
```
